```python
import numpy as np

kernel_name = "gpu_ag_gemm_m2048_k2048_n2048_bf16_none"


ABSMAX_RTOL = 0.008

COMPUTE_DTYPE = "bfloat16"
EPILOGUE = "none"
IN_QUANT = None
OUT_QUANT = None
IN_DTYPES = {"x": "bfloat16", "w_mat": "bfloat16"}
WORLD_SIZE = 8

def _to_bf16(x):
    u = np.ascontiguousarray(x, dtype=np.float32).view(np.uint32)
    return ((u + 0x7FFF + ((u >> 16) & 1)) & 0xFFFF0000).view(np.float32)

def setup_inputs():

    x = _to_bf16(np.random.randn(2048, 2048).astype(np.float32))
    w_mat = _to_bf16((np.random.randn(2048, 2048) / np.sqrt(2048)).astype(np.float32))
    return {"x": x, "w_mat": w_mat}

def _epilogue(y):
    return y

def reference(x, w_mat):
    y = (x @ w_mat).astype(np.float32)
    return _epilogue(y)
```

```cpp
#include <hip/hip_runtime.h>
#include <rccl/rccl.h>
#include <cstdio>

#define HIP_CHECK(expr) do { hipError_t _e = (expr); if (_e != hipSuccess) { \
    fprintf(stderr, "HIP error %s at %s:%d\n", hipGetErrorString(_e), __FILE__, __LINE__); \
    return; } } while (0)

typedef unsigned u32;
typedef unsigned long long u64;
typedef u32 u32x2 __attribute__((ext_vector_type(2)));
typedef u32 u32x4 __attribute__((ext_vector_type(4)));
typedef short bf16x8 __attribute__((ext_vector_type(8)));
typedef float f32x4 __attribute__((ext_vector_type(4)));
typedef __attribute__((address_space(1))) u32 gu32;

#define NPW 2
constexpr int NR = 8;
constexpr int MS = 256, KD = 2048, NS = 256;
constexpr u32 SHARD_BYTES = MS * KD * 2;
constexpr u32 UNIT = 16384, HALF = 8192;
constexpr u32 WS_FLAG = 0;
constexpr u32 WS_CNT = 65536;
constexpr u32 WS_TMO = WS_CNT + 256 * 128;
constexpr u32 WS_STAMP = 131072;
constexpr u32 WS_STAGE = 1u << 20;
constexpr u32 RECV_SRC = 8 * NPW * UNIT;
constexpr u32 WS_RECV = WS_STAGE + 2 * SHARD_BYTES;
constexpr u32 WS_END = WS_RECV + 2 * NR * RECV_SRC;
constexpr u64 DEADLINE = 100000000ull;

struct Params {
    const void* x;
    const void* w;
    void* out;
    char* ws[NR];
    int rank;
};

__device__ __forceinline__ __amdgpu_buffer_rsrc_t rsrc(const void* p, u32 bytes) {
    return __builtin_amdgcn_make_buffer_rsrc(const_cast<void*>(p), 0, bytes, 0x00020000);
}
#define SYS_LOAD(p) __hip_atomic_load((gu32*)(p), __ATOMIC_RELAXED, __HIP_MEMORY_SCOPE_SYSTEM)
#define SYS_STORE(p, v) __hip_atomic_store((gu32*)(p), (v), __ATOMIC_RELAXED, __HIP_MEMORY_SCOPE_SYSTEM)
#define VM0() asm volatile("s_waitcnt vmcnt(0)" ::: "memory")
#define STAMP(i) do {} while (0)

__global__ void __launch_bounds__(512, 2) ag_gemm(Params p) {
    __shared__ __attribute__((aligned(16))) f32x4 red[8 * 8 * 64 + 1];
    const int b = blockIdx.x, tid = threadIdx.x, lane = tid & 63;
    const int wave = __builtin_amdgcn_readfirstlane(tid >> 6);
    const int x = b & 7, idx = b >> 3;
    const int peer = (p.rank ^ x) & 7;
    char* me = p.ws[p.rank & 7];
    char* pw = p.ws[peer];
    const int c = lane & 15, g = lane >> 4;
    volatile __attribute__((address_space(3))) u32* inv_done =
        (volatile __attribute__((address_space(3))) u32*)&red[8 * 8 * 64];
    STAMP(0);

    const auto rx = rsrc(p.x, SHARD_BYTES);
    const u32 xoff = c * (KD * 2) + g * 16;
    u32x4 a[16];
    int duty = -1, hu = 0;
    if (wave == 0 && x != 0 && idx < 16 * NPW) {
        duty = 0;
        hu = ((idx >> 1) / NPW * 8 + (idx >> 1) % NPW) * 2 + (idx & 1);
    } else if (wave == 1 && b < 16 * (8 - NPW)) {
        duty = 1;
        hu = ((b >> 1) / (8 - NPW) * 8 + NPW + (b >> 1) % (8 - NPW)) * 2 + (b & 1);
    }
    if (duty >= 0) {
        const u32 so = (hu >> 4) * (32 * KD * 2) + (hu & 15) * 256;
#pragma unroll
        for (int j = 0; j < 4; ++j)
#pragma unroll
            for (int i = 0; i < 2; ++i)
                a[j * 2 + i] = __builtin_amdgcn_raw_buffer_load_b128(rx, xoff + j * 64, so + i * (16 * KD * 2), 0);
    }
    if (wave == 7) {
        if (lane == 0) *inv_done = 0;
        asm volatile("buffer_inv sc0 sc1" ::: "memory");
    }
    const u32 call = SYS_LOAD(me + WS_CNT + b * 128) + 1;
    const u32 par = __builtin_amdgcn_readfirstlane(call) & 1;
    if (duty >= 0) {
        const int rp = hu >> 4, w = (hu >> 1) & 7, h = hu & 1;
        char* dst = duty == 0 ? pw + WS_RECV + (par * NR + x) * RECV_SRC + (rp * NPW + w) * UNIT + h * HALF
                              : me + WS_STAGE + par * SHARD_BYTES + hu * HALF;
        const auto rd = rsrc(dst, HALF);
#pragma unroll
        for (int f = 0; f < 8; ++f)
            __builtin_amdgcn_raw_buffer_store_b128(a[f], rd, lane * 16, f * 1024, 17);
    }
    STAMP(1);
    asm volatile("s_waitcnt lgkmcnt(0)" ::: "memory");
    __builtin_amdgcn_s_barrier();
    if (duty >= 0) {
        VM0();
        const u32 fo = WS_FLAG + (hu >> 1) * 128 + (hu & 1) * 4;
        if (duty == 0) {
            if (lane == 0) SYS_STORE(pw + fo + x * 8192, call);
        } else {
#pragma unroll
            for (int q = 1; q < NR; ++q)
                if (lane == 0) SYS_STORE(p.ws[(p.rank ^ q) & 7] + fo + q * 8192, call);
        }
    }
    STAMP(2);

    const int rp = idx >> 2, n0 = (idx & 3) * 64;
    bf16x8 bf[8][4];
    {
        const auto rw = rsrc(p.w, KD * NS * 2);
        const u32 vo = g * 8 * (NS * 2) + c * 8 + n0 * 2;
#pragma unroll
        for (int j = 0; j < 8; ++j) {
            u32x2 raw[8];
#pragma unroll
            for (int e = 0; e < 8; ++e)
                raw[e] = __builtin_amdgcn_raw_buffer_load_b64(rw, vo + e * (NS * 2), (wave * 256 + j * 32) * (NS * 2), 0);
#pragma unroll
            for (int q = 0; q < 4; ++q) {
                u32x4 v;
#pragma unroll
                for (int t = 0; t < 4; ++t)
                    v[t] = __builtin_amdgcn_perm(raw[2 * t + 1][q >> 1], raw[2 * t][q >> 1],
                                                 (q & 1) ? 0x07060302u : 0x05040100u);
                bf[j][q] = __builtin_bit_cast(bf16x8, v);
            }
        }
    }
    if (wave == 7) {
        VM0();
        if (lane == 0) *inv_done = 1;
    }
    STAMP(3);

    bool bad = false;
    if (x == 0) {
        const u32 so = rp * (32 * KD * 2) + wave * 512;
#pragma unroll
        for (int j = 0; j < 8; ++j)
#pragma unroll
            for (int i = 0; i < 2; ++i)
                a[j * 2 + i] = __builtin_amdgcn_raw_buffer_load_b128(rx, xoff + j * 64, so + i * (16 * KD * 2), 0);
    } else {
        char* fp = lane < 2 ? me + WS_FLAG + x * 8192 + (rp * 8 + wave) * 128 + lane * 4 : me + WS_TMO;
        const u64 t0 = __builtin_amdgcn_s_memrealtime();
        for (;;) {
            const u32 v = SYS_LOAD(fp);
            if (__all(lane >= 2 || v >= call)) break;
            if (__any(lane >= 2 && v == call) || __builtin_amdgcn_s_memrealtime() - t0 > DEADLINE) {
                if (lane == 0) SYS_STORE(me + WS_TMO, call);
                bad = true;
                break;
            }
            __builtin_amdgcn_s_sleep(1);
        }
        while (*inv_done == 0) __builtin_amdgcn_s_sleep(1);
        asm volatile("" ::: "memory");
        STAMP(4);
        const u32 nb = bad ? 0u : UNIT;
        if (wave < NPW) {
            const auto ra = rsrc(me + WS_RECV + (par * NR + x) * RECV_SRC + (rp * NPW + wave) * UNIT, nb);
#pragma unroll
            for (int f = 0; f < 16; ++f)
                a[f] = __builtin_amdgcn_raw_buffer_load_b128(ra, lane * 16 + (f & 3) * 1024, (f >> 2) * 4096, 17);
        } else {
            const auto ra = rsrc(pw + WS_STAGE + par * SHARD_BYTES + (rp * 8 + wave) * UNIT, nb);
#pragma unroll
            for (int f = 0; f < 16; ++f)
                a[f] = __builtin_amdgcn_raw_buffer_load_b128(ra, lane * 16 + (f & 3) * 1024, (f >> 2) * 4096, 0);
        }
    }
    f32x4 acc[2][4];
#pragma unroll
    for (int i = 0; i < 2; ++i)
#pragma unroll
        for (int q = 0; q < 4; ++q) acc[i][q] = f32x4{0.f, 0.f, 0.f, 0.f};
#pragma unroll
    for (int j = 0; j < 8; ++j)
#pragma unroll
        for (int i = 0; i < 2; ++i)
#pragma unroll
            for (int q = 0; q < 4; ++q)
                acc[i][q] = __builtin_amdgcn_mfma_f32_16x16x32_bf16(
                    __builtin_bit_cast(bf16x8, a[j * 2 + i]), bf[j][q], acc[i][q], 0, 0, 0);
    STAMP(5);
    if (bad) acc[0][0][0] = acc[0][0][1] = acc[0][0][2] = acc[0][0][3] = __builtin_nanf("");

#pragma unroll
    for (int i = 0; i < 2; ++i)
#pragma unroll
        for (int r = 0; r < 4; ++r)
            red[(wave * 8 + i * 4 + r) * 64 + lane] = f32x4{acc[i][0][r], acc[i][1][r], acc[i][2][r], acc[i][3][r]};
    __syncthreads();
    f32x4 s = red[wave * 64 + lane];
#pragma unroll
    for (int v = 1; v < 8; ++v) s += red[(v * 8 + wave) * 64 + lane];
    typedef __bf16 bf16x4_t __attribute__((ext_vector_type(4)));
    const bf16x4_t o = __builtin_convertvector(s, bf16x4_t);
    const int row = peer * MS + rp * 32 + (wave >> 2) * 16 + g * 4 + (wave & 3);
    const auto ro = rsrc(p.out, NR * MS * NS * 2);
    __builtin_amdgcn_raw_buffer_store_b64(__builtin_bit_cast(u32x2, o), ro, row * (NS * 2) + (n0 + c * 4) * 2, 0, 0);
    if (tid == 0) SYS_STORE(me + WS_CNT + b * 128, call);
    STAMP(6);
}

extern "C" void kernel_launch(int rank, int nranks, ncclComm_t comm, hipStream_t stream,
                              void* const* d_in, const int* in_sizes, int n_in,
                              void* d_out, int out_size, const int* in_dtypes, int out_dtype,
                              void* const* d_out_all, void* const* d_ws_all, size_t ws_bytes) {
    (void)comm; (void)in_dtypes; (void)out_dtype; (void)d_out_all;
    if (nranks != NR || n_in != 2 || in_sizes[0] != MS * KD || in_sizes[1] != KD * NS ||
        out_size != NR * MS * NS || ws_bytes < WS_END || (unsigned)rank >= (unsigned)NR) {
        fprintf(stderr, "unexpected problem shape\n");
        return;
    }
    Params p;
    p.x = d_in[0];
    p.w = d_in[1];
    p.out = d_out;
    for (int i = 0; i < NR; ++i) p.ws[i] = (char*)d_ws_all[i];
    p.rank = rank;
    ag_gemm<<<256, 512, 0, stream>>>(p);
    HIP_CHECK(hipGetLastError());
}
```

```cpp
#include <hip/hip_runtime.h>
#include <rccl/rccl.h>
#include <cstdio>

#define HIP_CHECK(expr) do { hipError_t _e = (expr); if (_e != hipSuccess) { \
    fprintf(stderr, "HIP error %s at %s:%d\n", hipGetErrorString(_e), __FILE__, __LINE__); \
    return; } } while (0)

typedef unsigned u32;
typedef unsigned long long u64;
typedef u32 u32x2 __attribute__((ext_vector_type(2)));
typedef u32 u32x4 __attribute__((ext_vector_type(4)));
typedef short bf16x8 __attribute__((ext_vector_type(8)));
typedef float f32x4 __attribute__((ext_vector_type(4)));
typedef __attribute__((address_space(1))) u32 gu32;

#define WIN 3
#define POLL_SLEEP 4
#define T_Y 1230
#define T_R 1570
#define SPEC 1
constexpr int NR = 8;
constexpr int MS = 256, KD = 2048, NS = 256;
constexpr u32 SHARD_BYTES = MS * KD * 2;
constexpr u32 PH = 131072;
constexpr u32 LINK = 5 * PH;
constexpr u32 SLOT = 7 * LINK;
constexpr int NCLEAN = 12;
constexpr u32 WS_TMO = 0;
constexpr u32 WS_CNT = 8192;
constexpr u32 WS_RECV = 1u << 20;
constexpr u32 WS_END = WS_RECV + 3 * SLOT;
constexpr int RED_H = 260, RED_W = 520;
constexpr u32 DEADLINE = 50000000u;

constexpr u32 D_HELP = 1u << 31, D_PUSH = 1u << 30, D_SINGLE = 1u << 17;

struct Params {
    const void* x;
    const void* w;
    char* out[NR];
    char* ws[NR];
    int rank;
    u32 tab[256];
};

__device__ __forceinline__ __amdgpu_buffer_rsrc_t rsrc(const void* p, u32 bytes) {
    return __builtin_amdgcn_make_buffer_rsrc(const_cast<void*>(p), 0, bytes, 0x00020000);
}
#define SYS_LOAD(p) __hip_atomic_load((gu32*)(p), __ATOMIC_RELAXED, __HIP_MEMORY_SCOPE_SYSTEM)
#define SYS_STORE(p, v) __hip_atomic_store((gu32*)(p), (v), __ATOMIC_RELAXED, __HIP_MEMORY_SCOPE_SYSTEM)
#define VMCNT(n) asm volatile("s_waitcnt vmcnt(%0)" :: "n"(n) : "memory")
#define FENCE() asm volatile("" ::: "memory")
#define LD128(r, vo, so, aux) __builtin_amdgcn_raw_buffer_load_b128(r, vo, so, aux)
#define LD64(r, vo, so) __builtin_amdgcn_raw_buffer_load_b64(r, vo, so, 0)
#define NOW() __builtin_amdgcn_s_memrealtime()
#define SINCE(t0) ((u32)NOW() - (t0))
#define ANY(x) (__builtin_amdgcn_ballot_w64(x) != 0)
#define ALL(x) (__builtin_amdgcn_ballot_w64(!(x)) == 0)
#define STAMP(i) do {} while (0)
#define STAMP_DECL
#define STAMP_OUT() do {} while (0)
#define TS_ARG
#define TS_PASS

__device__ __forceinline__ void st128(u32x4 d, __amdgpu_buffer_rsrc_t r, u32 vo, u32 so) {
    __builtin_amdgcn_raw_buffer_store_b128(d, r, vo, so, 17);
    asm volatile("s_nop 1" :: "v"(d) : "memory");
}
__device__ __forceinline__ u32x4 nz(u32x4 v) {
#pragma unroll
    for (int i = 0; i < 4; ++i) v[i] = v[i] ? v[i] : 0x80000000u;
    return v;
}
__device__ __forceinline__ u32 min4(u32 m, u32x4 v) {
    return min(min(min(m, v[0]), v[1]), min(v[2], v[3]));
}
__device__ __forceinline__ u32x4 wfrag(const u32x2* raw, int q) {
    u32x4 v;
#pragma unroll
    for (int t = 0; t < 4; ++t)
        v[t] = __builtin_amdgcn_perm(raw[2 * t + 1][q >> 1], raw[2 * t][q >> 1], (q & 1) ? 0x07060302u : 0x05040100u);
    return v;
}
template <int S, int N>
__device__ __forceinline__ void push_steps(u32x4* d, __amdgpu_buffer_rsrc_t rd, u32 vd TS_ARG) {
    if constexpr (S < N) {
        const u32x4 t = nz(d[S]);
        if constexpr (S >= WIN) asm volatile("s_waitcnt vmcnt(%0)" :: "n"(WIN - 1), "v"(t) : "memory");
        st128(t, rd, vd, S * 32768);
        if constexpr ((S & 3) == 0) STAMP(1 + (S >> 2));
        push_steps<S + 1, N>(d, rd, vd TS_PASS);
    }
}

__device__ __forceinline__ u32 watch(char* addr, char* tmo, u32 call, u32 t0, int lane, bool& ok) {
    int nw = 16;
    while (ok) {
        const bool mine = lane < nw;
        const u32 v = SYS_LOAD(mine ? addr : tmo);
        const u32 seen = (u32)__builtin_amdgcn_ballot_w64(mine && v != 0);
        if (seen) return seen;
        if (ANY(!mine && v == call) || SINCE(t0) > DEADLINE) {
            SYS_STORE(tmo, call);
            ok = false;
        }
        nw = 4;
        __builtin_amdgcn_s_sleep(POLL_SLEEP);
    }
    return ~0u;
}

__global__ void __launch_bounds__(512, 2) ag_gemm(Params p) {
    __shared__ __attribute__((aligned(16))) f32x4 red[2][8 * RED_W];
    const int b = blockIdx.x, tid = threadIdx.x, lane = tid & 63;
    const int wave = __builtin_amdgcn_readfirstlane(tid >> 6);
    const u32 td = p.tab[b];
    char* me = p.ws[0];
    char* pws = p.ws[b & 7];
    const int c = lane & 15, g = lane >> 4;
    const u32 lane16 = lane * 16;
    STAMP_DECL;
    STAMP(0);
    const u32 t0 = (u32)NOW();
    const auto rx = rsrc(p.x, SHARD_BYTES);
    const auto rw = rsrc(p.w, SHARD_BYTES);
    const u32 xoff = c * (KD * 2) + g * 16;
    const u32 woff = g * 8 * (NS * 2) + c * 8;
    char* cptr = me + WS_CNT + b * 128 + wave * 4;
    u32 cnt = *(const __attribute__((address_space(4))) u32*)cptr;
    u32 call, slot;
#define GET_CALL() do { asm volatile("" : "+s"(cnt), "+s"(pws) :: "memory"); call = cnt + 1; slot = call % 3; } while (0)
#define PUT_CALL() do { if (lane == 0) SYS_STORE(cptr, call); } while (0)

    if (td & D_HELP) {
        const int x = b & 7, hi = td & 15;
        if (!(td & D_PUSH)) {
            GET_CALL();
            PUT_CALL();
            const auto rz = rsrc(me + WS_RECV + ((call + 2) % 3) * SLOT, SLOT);
#pragma unroll 1
            for (u32 kb = hi * 8 + wave; kb < SLOT / 1024; kb += NCLEAN * 8)
                st128(u32x4{0u, 0u, 0u, 0u}, rz, lane16, kb * 1024);
        } else if (wave < 2) {
            const int pw = hi * 2 + wave;
            const u32 sx = (pw & 1) * (16 * KD * 2) + (pw >> 2) * 512 + ((pw >> 1) & 1) * 64;
            const u32 doff = WS_RECV + (x - 1) * LINK + pw * 1024;
            u32x4 d[20];
            if (x < 4) {
#pragma unroll
                for (int s = 0; s < 3; ++s) d[s] = LD128(rx, xoff + s * 128, sx, 0);
                GET_CALL();
                const auto rd = rsrc(pws + doff + slot * SLOT, LINK - pw * 1024);
                push_steps<0, 3>(d, rd, lane16 TS_PASS);
                PUT_CALL();
#pragma unroll
                for (int s = 3; s < 20; ++s)
                    d[s] = LD128(rx, xoff + (s & 3) * 128, sx + (s >> 2) * (32 * KD * 2), 0);
                FENCE();
                push_steps<3, 20>(d, rd, lane16 TS_PASS);
            } else {
                u32x2 t[16];
#pragma unroll
                for (int e = 0; e < 8; ++e)
                    t[e] = LD64(rw, woff + e * (NS * 2), (x & 3) * 128 + pw * (32 * NS * 2));
                GET_CALL();
                const auto rd = rsrc(pws + doff + slot * SLOT, LINK - pw * 1024);
#pragma unroll
                for (int s = 0; s < 4; ++s) d[s] = wfrag(t, s);
                push_steps<0, 3>(d, rd, lane16 TS_PASS);
                PUT_CALL();
#pragma unroll
                for (int e = 8; e < 16; ++e)
                    t[e] = LD64(rw, woff + (e & 7) * (NS * 2), (x & 3) * 128 + (32 + pw) * (32 * NS * 2));
#pragma unroll
                for (int s = 8; s < 20; ++s)
                    d[s] = LD128(rx, xoff + (s & 3) * 128, sx + ((s >> 2) + 3) * (32 * KD * 2), 0);
                FENCE();
#pragma unroll
                for (int s = 4; s < 8; ++s) d[s] = wfrag(t + 8, s & 3);
                push_steps<3, 4>(d, rd, lane16 TS_PASS);
                __builtin_amdgcn_s_barrier();
                push_steps<4, 20>(d, rd, lane16 TS_PASS);
            }
        } else {
            GET_CALL();
            PUT_CALL();
            if (x >= 4) {
                const auto ro = rsrc(p.out[0], SHARD_BYTES);
                u32 off[2];
#pragma unroll
                for (int i = 0; i < 2; ++i) {
                    const int id = hi + 16 * i, j = (id - 20) / 3;
                    const int u = id < 20 ? id / 5 : 4 + j + (4 + j >= x), r = id < 20 ? id % 5 : 5 + (id - 20) % 3;
                    off[i] = (((p.rank ^ x ^ u) & 7) * MS + r * 32 + (wave - 2) * 8 + (lane >> 3)) * (NS * 2) + (x & 3) * 128 + (lane & 7) * 16;
                }
                if (hi >= 13) off[1] = off[0];
                if (wave < 6) {
                    st128(u32x4{0u, 0u, 0u, 0u}, ro, off[0], 0);
                    st128(u32x4{0u, 0u, 0u, 0u}, ro, off[1], 0);
                    VMCNT(0);
                }
                __builtin_amdgcn_s_barrier();
                if (wave < 6) {
                    while (SINCE(t0) < T_R) __builtin_amdgcn_s_sleep(2);
                    for (u32 it = 1;; ++it) {
                        const u32x4 y0 = LD128(ro, off[0], 0, 17), y1 = LD128(ro, off[1], 0, 17);
                        if (ALL(min4(min4(~0u, y0), y1) != 0)) break;
                        if ((it & 255) == 0 && SINCE(t0) > DEADLINE + DEADLINE / 2) {
                            SYS_STORE(me + WS_TMO, call);
                            st128(u32x4{0x7FC07FC0u, 0x7FC07FC0u, 0x7FC07FC0u, 0x7FC07FC0u}, ro, off[0], 0);
                            break;
                        }
                    }
                    STAMP(2);
                }
            }
        }
        STAMP_OUT();
        return;
    }

    const int v = (td >> 12) & 7, q = (td >> 15) & 3;
    const int u0 = td & 7, r0 = (td >> 3) & 7, u1 = (td >> 6) & 7, r1 = (td >> 9) & 7;
    char* optr = p.out[v];
    int rank = p.rank;
    bool ok = true;
    bf16x8 bf[8][4];
    u32x4 a[16];
    f32x4 acc[2][4];
#define A_OFF(f) (((f) >> 2) * 32768 + ((f) & 3) * 1024)
#define B_OFF(f) (((f) & 3) * 32768 + ((f) >> 2) * 1024)
#define A_BASE(u, r) (me + WS_RECV + slot * SLOT + ((u) - 1) * LINK + ((u) < 4 ? (r) : (r) - 3) * PH + wave * 4096)
#define MMA(f) do { _Pragma("unroll") for (int n = 0; n < 4; ++n) \
            acc[(f) & 1][n] = __builtin_amdgcn_mfma_f32_16x16x32_bf16( \
                __builtin_bit_cast(bf16x8, a[f]), bf[(f) >> 1][n], acc[(f) & 1][n], 0, 0, 0); } while (0)
#define PUT(h) do { if (!ok) { SYS_STORE(me + WS_TMO, call); \
            acc[0][0] = acc[1][0] = f32x4{__builtin_nanf(""), __builtin_nanf(""), __builtin_nanf(""), __builtin_nanf("")}; } \
        _Pragma("unroll") for (int i = 0; i < 2; ++i) _Pragma("unroll") for (int rr = 0; rr < 4; ++rr) \
            red[h][wave * RED_W + (c & 1) * RED_H + (i * 16 + g * 4 + rr) * 8 + (c >> 1)] = \
                f32x4{acc[i][0][rr], acc[i][1][rr], acc[i][2][rr], acc[i][3][rr]}; } while (0)

    if (u0 == 0) {
#pragma unroll
        for (int f = 0; f < 16; ++f)
            a[f] = LD128(rx, xoff, r0 * (32 * KD * 2) + wave * 512 + (f >> 1) * 64 + (f & 1) * (16 * KD * 2), 0);
    } else {
#pragma unroll
        for (int f = 0; f < 16; ++f) asm volatile("" : "=v"(a[f]));
    }
    if (v == 0) {
        u32x2 raw[64];
#pragma unroll
        for (int n = 0; n < 64; ++n)
            raw[n] = LD64(rw, woff + q * 128 + (n & 7) * (NS * 2), (wave * 8 + (n >> 3)) * (32 * NS * 2));
#pragma unroll
        for (int j = 0; j < 8; ++j) {
#pragma unroll
            for (int n = 0; n < 4; ++n) bf[j][n] = __builtin_bit_cast(bf16x8, wfrag(raw + j * 8, n));
            asm volatile("" : "+v"(bf[j][0]), "+v"(bf[j][1]), "+v"(bf[j][2]), "+v"(bf[j][3]) :: "memory");
        }
    } else {
#pragma unroll
        for (int j = 0; j < 8; ++j) asm volatile("" : "=v"(bf[j][0]), "=v"(bf[j][1]), "=v"(bf[j][2]), "=v"(bf[j][3]));
    }
    GET_CALL();
    PUT_CALL();
    STAMP(1);
    if (v | u0) {
        char* bbase = me + WS_RECV + slot * SLOT + (v - 1) * LINK + (wave >> 2) * PH + (wave & 3) * 8192;
        char* abase = A_BASE(u0, r0);
        const auto rb = rsrc(bbase, PH - (wave & 3) * 8192), ra = rsrc(abase, PH - wave * 4096);
        const u64 all = (v ? 0x11111111ull : 0) | (u0 ? 0x1111ull << 32 : 0);
        char* can = (lane < 32 ? bbase + (lane >> 3) * 32768 + (lane & 7) * 1024 : abase + A_OFF(lane & 15)) + 1020;
        u64 todo = all, look = all * 15;
#pragma unroll 1
        for (;;) {
            const bool mine = (look >> lane) & 1;
            const u32 w = SYS_LOAD(mine ? can : me + WS_TMO);
            const u64 seen = __builtin_amdgcn_ballot_w64(mine && w != 0);
            u64 get = seen & (seen >> 1);
            get &= (get >> 2) & todo;
            if (ANY(!mine && w == call) || SINCE(t0) > DEADLINE) { ok = false; get = todo; }
#pragma unroll
            for (int G = 0; G < 8; ++G)
                if ((get >> (4 * G)) & 1) {
#pragma unroll
                    for (int i = 0; i < 4; ++i)
                        bf[(G & 1) * 4 + i][G >> 1] = __builtin_bit_cast(bf16x8, LD128(rb, lane16 + i * 1024, (G >> 1) * 32768 + (G & 1) * 4096, 17));
                }
#pragma unroll
            for (int G = 0; G < 4; ++G)
                if ((get >> (32 + 4 * G)) & 1) {
#pragma unroll
                    for (int i = 0; i < 4; ++i) a[G * 4 + i] = LD128(ra, lane16 + i * 1024, G * 32768, 17);
                }
            todo &= ~get;
            if (!todo) break;
            look = todo * 15;
            if (!seen) {
                if (todo == all) look = todo & 0x100000011ull;
                __builtin_amdgcn_s_sleep(POLL_SLEEP);
            }
        }
#pragma unroll 1
        for (;;) {
            u32 mb = ~0u, ma = ~0u;
#pragma unroll
            for (int f = 0; f < 32; ++f) mb = min4(mb, __builtin_bit_cast(u32x4, bf[f >> 2][f & 3]));
#pragma unroll
            for (int f = 0; f < 16; ++f) ma = min4(ma, a[f]);
            const bool rb_ = v && ANY(mb == 0), ra_ = u0 && ANY(ma == 0);
            if (__builtin_expect(!(rb_ || ra_) || !ok, 1)) break;
            if (rb_) {
#pragma unroll
                for (int f = 0; f < 32; ++f) bf[f >> 2][f & 3] = __builtin_bit_cast(bf16x8, LD128(rb, lane16, B_OFF(f), 17));
            }
            if (ra_) {
#pragma unroll
                for (int f = 0; f < 16; ++f) a[f] = LD128(ra, lane16, A_OFF(f), 17);
            }
            if (SINCE(t0) > DEADLINE) ok = false;
        }
    }
    asm volatile("" : "+s"(optr), "+s"(rank) :: "memory");
    STAMP(2);
#pragma unroll
    for (int i = 0; i < 2; ++i)
#pragma unroll
        for (int n = 0; n < 4; ++n) acc[i][n] = f32x4{0.f, 0.f, 0.f, 0.f};
#pragma unroll
    for (int f = 0; f < 16; ++f) MMA(f);
    PUT(0);
    STAMP(3);

    bool late = v == 0;
    if (!(td & D_SINGLE)) {
        char* abase = A_BASE(u1, r1);
        const u32 coff = A_OFF(lane & 15) + 1020;
        const u32x4 ra = {(u32)(u64)abase, (u32)((u64)abase >> 32) & 0xFFFFu, PH - wave * 4096, 0x00020000u};
        u32 seen = watch(abase + coff, me + WS_TMO, call, t0, lane, ok), pend = 0x1111, cur = 0, w = 0;
        u32 lim = ok ? 1 : 0;
        STAMP(5);
#pragma unroll
        for (int i = 0; i < 2; ++i) {
#pragma unroll
            for (int n = 0; n < 4; ++n) acc[i][n] = f32x4{0.f, 0.f, 0.f, 0.f};
            asm volatile("" : "+v"(acc[i][0]), "+v"(acc[i][1]), "+v"(acc[i][2]), "+v"(acc[i][3]), "+s"(pend), "+s"(cur));
        }
#pragma unroll 1
        for (u32 it = 1;; ++it) {
            const u32 rest = pend & ~cur;
            u32 nxt = seen & (seen >> 1);
            nxt &= (nxt >> 2) & rest;
            if (__builtin_popcount(rest & ~nxt) <= SPEC) nxt = rest;
#pragma unroll
            for (int t = 0; t < 4; ++t)
                if ((nxt >> (4 * t)) & 1)
                    asm volatile("buffer_load_dwordx4 %0, %4, %5, %6 offen sc0 sc1\n\t"
                                 "buffer_load_dwordx4 %1, %4, %5, %6 offen offset:1024 sc0 sc1\n\t"
                                 "buffer_load_dwordx4 %2, %4, %5, %6 offen offset:2048 sc0 sc1\n\t"
                                 "buffer_load_dwordx4 %3, %4, %5, %6 offen offset:3072 sc0 sc1"
                                 : "+v"(a[4 * t]), "+v"(a[4 * t + 1]), "+v"(a[4 * t + 2]), "+v"(a[4 * t + 3])
                                 : "v"(lane16), "s"(ra), "s"(t * 32768) : "memory");
            if (rest != nxt)
                asm volatile("buffer_load_dword %0, %1, %2, 0 offen sc0 sc1" : "+v"(w) : "v"(coff), "s"(ra) : "memory");
#pragma unroll
            for (int t = 0; t < 4; ++t)
                if ((cur >> (4 * t)) & 1) {
                    u32 m = ~0u;
#pragma unroll
                    for (int f = 0; f < 4; ++f) m = min4(m, a[4 * t + f]);
                    if (!ANY(m < lim)) {
                        MMA(4 * t); MMA(4 * t + 1); MMA(4 * t + 2); MMA(4 * t + 3);
                        pend &= ~(1u << (4 * t));
                    }
                }
            if (!pend) break;
            if (!late) late = SINCE(t0) >= T_Y;
            cur = nxt;
            asm volatile("s_waitcnt vmcnt(0)"
                         : "+v"(a[0]), "+v"(a[1]), "+v"(a[2]), "+v"(a[3]), "+v"(a[4]), "+v"(a[5]), "+v"(a[6]), "+v"(a[7]),
                           "+v"(a[8]), "+v"(a[9]), "+v"(a[10]), "+v"(a[11]), "+v"(a[12]), "+v"(a[13]), "+v"(a[14]),
                           "+v"(a[15]), "+v"(w) :: "memory");
            seen = (u32)__builtin_amdgcn_ballot_w64(w != 0);
            if ((it & 255) == 0 && SINCE(t0) > DEADLINE) { ok = false; lim = 0; seen = 0xFFFF; }
        }
        PUT(1);
        STAMP(6);
    }
    __syncthreads();
    const int h = wave < 4;
    if (!(h && (td & D_SINGLE))) {
        const int t = tid & 255;
        const f32x4* rs = red[h] + t;
        f32x4 s0 = rs[0], s1 = rs[RED_H];
#pragma unroll
        for (int n = 1; n < 8; ++n) { s0 += rs[n * RED_W]; s1 += rs[n * RED_W + RED_H]; }
        typedef __bf16 bf16x4_t __attribute__((ext_vector_type(4)));
        const u32x2 o0 = __builtin_bit_cast(u32x2, __builtin_convertvector(s0, bf16x4_t));
        const u32x2 o1 = __builtin_bit_cast(u32x2, __builtin_convertvector(s1, bf16x4_t));
        u32x4 o = {o0[0], o0[1], o1[0], o1[1]};
        const u32 ooff = (((rank ^ (h ? u1 : u0)) & 7) * MS + (h ? r1 : r0) * 32 + (t >> 3)) * (NS * 2) + q * 128 + (t & 7) * 16;
        if (v) {
            o = nz(o);
            if (!late) while (SINCE(t0) < T_Y) __builtin_amdgcn_s_sleep(2);
        }
        __builtin_amdgcn_raw_buffer_store_b128(o, rsrc(optr, SHARD_BYTES), ooff, 0, 17);
    }
    STAMP(4);
    STAMP_OUT();
}

static void build_table(u32* tab) {
    u32 list[8][32];
    int n[8] = {0};
    auto add = [&](int x, u32 d) { list[x][n[x]++] = d; };
    auto ur = [](int u, int r) { return (u32)(u | r << 3); };
    auto pair = [](const u32* e, int m, int i) { return 2 * i + 1 == m ? (e[i] | D_SINGLE) : (e[i] | e[(m + 1) / 2 + i] << 6); };
    for (int x = 1; x < 8; ++x)
        for (int i = 0; i < 16; ++i) add(x, D_HELP | D_PUSH | i);
    for (int i = 0; i < NCLEAN; ++i) add(0, D_HELP | i);
    const int S[4][5][2] = {{{0, 0}, {0, 1}, {1, 2}, {1, 4}, {2, 4}}, {{0, 2}, {1, 0}, {1, 1}, {1, 3}, {3, 4}},
                            {{0, 3}, {2, 0}, {2, 1}, {2, 2}, {2, 3}}, {{0, 4}, {3, 0}, {3, 1}, {3, 2}, {3, 3}}};
    for (int x = 4; x < 8; ++x)
        for (int v = 4; v < 8; ++v) {
            u32 e[8];
            int m = 0;
            for (int ph = -1; ph < 5; ++ph) {
                for (int i = 0; i < 5; ++i) {
                    const int u = S[x - 4][i][0], r = S[x - 4][i][1];
                    if ((u ? r : -1) == ph) e[m++] = ur(u, r);
                }
                if (ph >= 2 && x != v) e[m++] = ur(x, ph + 3);
            }
            for (int i = 0; i < (m + 1) / 2; ++i) add(x, pair(e, m, i) | v << 12 | (v & 3) << 15);
        }
    {
        u32 e[35];
        int m = 0;
        for (int r = 0; r < 8; ++r) e[m++] = ur(0, r);
        for (int ph = 0; ph < 5; ++ph) {
            for (int u = 1; u < 4; ++u) e[m++] = ur(u, ph);
            if (ph >= 2) for (int u = 4; u < 8; ++u) e[m++] = ur(u, ph + 3);
        }
        for (int i = 0; i < 18; ++i)
            for (int q = 0; q < 4; ++q) add(i == 17 ? 4 + q : i == 16 ? 0 : i & 3, pair(e, m, i) | q << 15);
    }
    for (int x = 0; x < 8; ++x) for (int i = 0; i < 32; ++i) tab[i * 8 + x] = list[x][i];
}

extern "C" void kernel_launch(int rank, int nranks, ncclComm_t comm, hipStream_t stream,
                              void* const* d_in, const int* in_sizes, int n_in,
                              void* d_out, int out_size, const int* in_dtypes, int out_dtype,
                              void* const* d_out_all, void* const* d_ws_all, size_t ws_bytes) {
    (void)comm; (void)in_dtypes; (void)out_dtype; (void)d_out;
    if (nranks != NR || n_in != 2 || in_sizes[0] != MS * KD || in_sizes[1] != KD * NS ||
        out_size != NR * MS * NS || ws_bytes < WS_END || (unsigned)rank >= (unsigned)NR) {
        fprintf(stderr, "unexpected problem shape\n");
        return;
    }
    Params p;
    p.x = d_in[0];
    p.w = d_in[1];
    for (int i = 0; i < NR; ++i) {
        p.out[i] = (char*)d_out_all[rank ^ i];
        p.ws[i] = (char*)d_ws_all[rank ^ i];
    }
    p.rank = rank;
    build_table(p.tab);
    ag_gemm<<<256, 512, 0, stream>>>(p);
    HIP_CHECK(hipGetLastError());
}
```

```cpp
#include <hip/hip_runtime.h>
#include <rccl/rccl.h>
#include <cstdio>

#define HIP_CHECK(expr) do { hipError_t _e = (expr); if (_e != hipSuccess) { \
    fprintf(stderr, "HIP error %s at %s:%d\n", hipGetErrorString(_e), __FILE__, __LINE__); \
    return; } } while (0)

typedef unsigned u32;
typedef unsigned long long u64;
typedef u32 u32x2 __attribute__((ext_vector_type(2)));
typedef u32 u32x4 __attribute__((ext_vector_type(4)));
typedef short bf16x8 __attribute__((ext_vector_type(8)));
typedef float f32x4 __attribute__((ext_vector_type(4)));
typedef __attribute__((address_space(1))) u32 gu32;

#define TAU 61
#define TAUF 62
#define TZERO -30
#define T_Y 1230
#define T_R 1495
#define COLD_TH 140
#define COLD_BASE 90
#define POLL_SLEEP 4
#define SPEC 1
constexpr int NR = 8;
constexpr int MS = 256, KD = 2048, NS = 256;
constexpr u32 SHARD_BYTES = MS * KD * 2;
constexpr u32 PH = 131072;
constexpr u32 LINK = 5 * PH;
constexpr u32 LSTRIDE = 2u << 20;
constexpr int NCLEAN = 12;
constexpr u32 WS_TMO = 0;
constexpr u32 WS_CNT = 8192;
constexpr u32 WS_RECV = 2u << 20;
constexpr u32 WS_END = WS_RECV + 7 * LSTRIDE;
constexpr int RED_H = 260, RED_W = 520;
constexpr u32 DEADLINE = 50000000u;

constexpr u32 D_HELP = 1u << 31, D_PUSH = 1u << 30, D_SINGLE = 1u << 17;

struct Params {
    const void* x;
    const void* w;
    char* out[NR];
    char* ws[NR];
    int rank;
    u32 tab[256];
};

__device__ __forceinline__ __amdgpu_buffer_rsrc_t rsrc(const void* p, u32 bytes) {
    return __builtin_amdgcn_make_buffer_rsrc(const_cast<void*>(p), 0, bytes, 0x00020000);
}
#define SYS_LOAD(p) __hip_atomic_load((gu32*)(p), __ATOMIC_RELAXED, __HIP_MEMORY_SCOPE_SYSTEM)
#define SYS_STORE(p, v) __hip_atomic_store((gu32*)(p), (v), __ATOMIC_RELAXED, __HIP_MEMORY_SCOPE_SYSTEM)
#define VMCNT(n) asm volatile("s_waitcnt vmcnt(%0)" :: "n"(n) : "memory")
#define FENCE() asm volatile("" ::: "memory")
#define LD128(r, vo, so, aux) __builtin_amdgcn_raw_buffer_load_b128(r, vo, so, aux)
#define LD64(r, vo, so) __builtin_amdgcn_raw_buffer_load_b64(r, vo, so, 0)
#define NOW() __builtin_amdgcn_s_memrealtime()
#define SINCE(t0) ((u32)NOW() - (t0))
#define ANY(x) (__builtin_amdgcn_ballot_w64(x) != 0)
#define ALL(x) (__builtin_amdgcn_ballot_w64(!(x)) == 0)

__device__ __forceinline__ void st128(u32x4 d, __amdgpu_buffer_rsrc_t r, u32 vo, u32 so) {
    __builtin_amdgcn_raw_buffer_store_b128(d, r, vo, so, 17);
    asm volatile("s_nop 1" :: "v"(d) : "memory");
}
__device__ __forceinline__ u32x4 nz(u32x4 v) {
#pragma unroll
    for (int i = 0; i < 4; ++i) v[i] = v[i] ? v[i] : 0x80000000u;
    return v;
}
__device__ __forceinline__ u32 min4(u32 m, u32x4 v) {
    return min(min(min(m, v[0]), v[1]), min(v[2], v[3]));
}
__device__ __forceinline__ u32x4 wfrag(const u32x2* raw, int q) {
    u32x4 v;
#pragma unroll
    for (int t = 0; t < 4; ++t)
        v[t] = __builtin_amdgcn_perm(raw[2 * t + 1][q >> 1], raw[2 * t][q >> 1], (q & 1) ? 0x07060302u : 0x05040100u);
    return v;
}
__device__ __forceinline__ u32 origin(u32 t0, u32 ts) { return ts - t0 > COLD_TH ? ts - COLD_BASE : t0; }
constexpr int slot_of(int s, bool far) { return !far ? s : s < 3 ? 8 + s : s < 11 ? s - 3 : s; }
template <int S, int N, bool FAR = false>
__device__ __forceinline__ void push_steps(u32x4* d, __amdgpu_buffer_rsrc_t rd, u32 vd, u32 t0) {
    if constexpr (S < N) {
        const u32x4 t = nz(d[slot_of(S, FAR)]);
        if constexpr (S >= 3) asm volatile("s_waitcnt vmcnt(%0)" :: "n"(S < 6 ? (FAR ? 20 : 17) : 2), "v"(t) : "memory");
        if constexpr (S >= 4) while (SINCE(t0) < (u32)(TZERO + S * (FAR ? TAUF : TAU))) __builtin_amdgcn_s_sleep(1);
        st128(t, rd, vd, slot_of(S, FAR) * 32768);
        push_steps<S + 1, N, FAR>(d, rd, vd, t0);
    }
}

__device__ __forceinline__ u32 watch(char* addr, char* tmo, u32 call, u32 t0, int lane, bool& ok) {
    int nw = 16;
    while (ok) {
        const bool mine = lane < nw;
        const u32 v = SYS_LOAD(mine ? addr : tmo);
        const u32 seen = (u32)__builtin_amdgcn_ballot_w64(mine && v != 0);
        if (seen) return seen;
        if (ANY(!mine && v == call) || SINCE(t0) > DEADLINE) {
            SYS_STORE(tmo, call);
            ok = false;
        }
        nw = 4;
        __builtin_amdgcn_s_sleep(POLL_SLEEP);
    }
    return ~0u;
}

__global__ void __launch_bounds__(512, 2) ag_gemm(Params p) {
    __shared__ __attribute__((aligned(16))) f32x4 red[2][8 * RED_W];
    const int b = blockIdx.x, tid = threadIdx.x, lane = tid & 63;
    const int wave = __builtin_amdgcn_readfirstlane(tid >> 6);
    const u32 td = p.tab[b];
    char* me = p.ws[0];
    char* pws = p.ws[b & 7];
    const int c = lane & 15, g = lane >> 4;
    const u32 lane16 = lane * 16;
    const u32 t0 = (u32)NOW();
    const auto rx = rsrc(p.x, SHARD_BYTES);
    const auto rw = rsrc(p.w, SHARD_BYTES);
    const u32 xoff = c * (KD * 2) + g * 16;
    const u32 woff = g * 8 * (NS * 2) + c * 8;
    char* cptr = me + WS_CNT + b * 128 + wave * 4;
    u32 cnt = *(const __attribute__((address_space(4))) u32*)cptr;
    u32 call, slot;
#define GET_CALL() do { asm volatile("" : "+s"(cnt), "+s"(pws) :: "memory"); call = cnt + 1; slot = call % 3; } while (0)
#define PUT_CALL() do { if (lane == 0) SYS_STORE(cptr, call); } while (0)

    if (__builtin_expect(td & D_HELP, 1)) {
        const int x = b & 7, hi = td & 15;
        if (__builtin_expect(!(td & D_PUSH), 0)) {
            GET_CALL();
            PUT_CALL();
            const auto rz = rsrc(me + WS_RECV + ((call + 2) % 3) * LINK, 6 * LSTRIDE + LINK);
#pragma unroll 1
            for (u32 kb = hi * 8 + wave; kb < 7 * LINK / 1024; kb += NCLEAN * 8)
                st128(u32x4{0u, 0u, 0u, 0u}, rz, lane16, kb / (LINK / 1024) * LSTRIDE + kb % (LINK / 1024) * 1024);
        } else if (__builtin_expect(wave < 2, 1)) {
            const int pw = hi * 2 + wave;
            const u32 sx = (pw & 1) * (16 * KD * 2) + (pw >> 2) * 512 + ((pw >> 1) & 1) * 64;
            const u32 doff = WS_RECV + (x - 1) * LSTRIDE + pw * 1024;
            u32x4 d[20];
            if (x < 4) {
#pragma unroll
                for (int s = 0; s < 6; ++s) d[s] = LD128(rx, xoff + (s & 3) * 128, sx + (s >> 2) * (32 * KD * 2), 0);
                GET_CALL();
                const auto rd = rsrc(pws + doff + slot * LINK, LINK - pw * 1024);
                push_steps<0, 3>(d, rd, lane16, t0);
                const u32 ts = (u32)NOW();
                PUT_CALL();
#pragma unroll
                for (int s = 6; s < 20; ++s)
                    d[s] = LD128(rx, xoff + (s & 3) * 128, sx + (s >> 2) * (32 * KD * 2), 0);
                FENCE();
                push_steps<3, 20>(d, rd, lane16, origin(t0, ts));
            } else {
                u32x2 t[16];
#pragma unroll
                for (int s = 8; s < 11; ++s) d[s] = LD128(rx, xoff + (s & 3) * 128, sx + 5 * (32 * KD * 2), 0);
#pragma unroll
                for (int e = 0; e < 8; ++e)
                    t[e] = LD64(rw, woff + e * (NS * 2), (x & 3) * 128 + pw * (32 * NS * 2));
                GET_CALL();
                const auto rd = rsrc(pws + doff + slot * LINK, LINK - pw * 1024);
                push_steps<0, 3, true>(d, rd, lane16, t0);
                const u32 ts = (u32)NOW();
#pragma unroll
                for (int s = 0; s < 4; ++s) d[s] = wfrag(t, s);
                PUT_CALL();
#pragma unroll
                for (int e = 8; e < 16; ++e)
                    t[e] = LD64(rw, woff + (e & 7) * (NS * 2), (x & 3) * 128 + (32 + pw) * (32 * NS * 2));
#pragma unroll
                for (int s = 11; s < 20; ++s)
                    d[s] = LD128(rx, xoff + (s & 3) * 128, sx + ((s >> 2) + 3) * (32 * KD * 2), 0);
                FENCE();
                const u32 t1 = origin(t0, ts);
                push_steps<3, 7, true>(d, rd, lane16, t1);
                __builtin_amdgcn_sched_barrier(0);
#pragma unroll
                for (int s = 4; s < 8; ++s) d[s] = wfrag(t + 8, s & 3);
                __builtin_amdgcn_s_barrier();
                push_steps<7, 20, true>(d, rd, lane16, t1);
            }
        } else {
            if (wave == 2 && lane == 0) SYS_STORE(pws + WS_RECV + x * LSTRIDE - 128 * (hi + 1), 0u);
            GET_CALL();
            PUT_CALL();
            if (x >= 4) {
                const auto ro = rsrc(p.out[0], SHARD_BYTES);
                u32 off[2];
#pragma unroll
                for (int i = 0; i < 2; ++i) {
                    const int id = hi + 16 * i, j = (id - 20) / 3;
                    const int u = id < 20 ? id / 5 : 4 + j + (4 + j >= x), r = id < 20 ? id % 5 : 5 + (id - 20) % 3;
                    off[i] = (((p.rank ^ x ^ u) & 7) * MS + r * 32 + (wave - 2) * 8 + (lane >> 3)) * (NS * 2) + (x & 3) * 128 + (lane & 7) * 16;
                }
                if (hi >= 13) off[1] = off[0];
                if (wave < 6) {
                    st128(u32x4{0u, 0u, 0u, 0u}, ro, off[0], 0);
                    st128(u32x4{0u, 0u, 0u, 0u}, ro, off[1], 0);
                    VMCNT(0);
                }
                __builtin_amdgcn_s_barrier();
                if (wave == 7 && hi < 4) {
                    const u32 t = SYS_LOAD(p.out[4 + hi]);
                    asm volatile("" :: "v"(t));
                }
                if (wave < 6) {
                    while (SINCE(t0) < T_R) __builtin_amdgcn_s_sleep(2);
                    for (u32 it = 1;; ++it) {
                        const u32x4 y0 = LD128(ro, off[0], 0, 17), y1 = LD128(ro, off[1], 0, 17);
                        if (ALL(min4(min4(~0u, y0), y1) != 0)) break;
                        if ((it & 255) == 0 && SINCE(t0) > DEADLINE + DEADLINE / 2) {
                            SYS_STORE(me + WS_TMO, call);
                            st128(u32x4{0x7FC07FC0u, 0x7FC07FC0u, 0x7FC07FC0u, 0x7FC07FC0u}, ro, off[0], 0);
                            break;
                        }
                    }
                }
            }
        }
        return;
    }

    const int v = (td >> 12) & 7, q = (td >> 15) & 3;
    const int u0 = td & 7, r0 = (td >> 3) & 7, u1 = (td >> 6) & 7, r1 = (td >> 9) & 7;
    char* optr = p.out[v];
    int rank = p.rank;
    bool ok = true;
    bf16x8 bf[8][4];
    u32x4 a[16];
    f32x4 acc[2][4];
#define A_OFF(f) (((f) >> 2) * 32768 + ((f) & 3) * 1024)
#define B_OFF(f) (((f) & 3) * 32768 + ((f) >> 2) * 1024)
#define A_BASE(u, r) (me + WS_RECV + slot * LINK + ((u) - 1) * LSTRIDE + ((u) < 4 ? (r) : (r) - 3) * PH + wave * 4096)
#define MMA(f) do { _Pragma("unroll") for (int n = 0; n < 4; ++n) \
            acc[(f) & 1][n] = __builtin_amdgcn_mfma_f32_16x16x32_bf16( \
                __builtin_bit_cast(bf16x8, a[f]), bf[(f) >> 1][n], acc[(f) & 1][n], 0, 0, 0); } while (0)
#define PUT(h) do { if (!ok) { SYS_STORE(me + WS_TMO, call); \
            acc[0][0] = acc[1][0] = f32x4{__builtin_nanf(""), __builtin_nanf(""), __builtin_nanf(""), __builtin_nanf("")}; } \
        _Pragma("unroll") for (int i = 0; i < 2; ++i) _Pragma("unroll") for (int rr = 0; rr < 4; ++rr) \
            red[h][wave * RED_W + (c & 1) * RED_H + (i * 16 + g * 4 + rr) * 8 + (c >> 1)] = \
                f32x4{acc[i][0][rr], acc[i][1][rr], acc[i][2][rr], acc[i][3][rr]}; } while (0)

    if (u0 == 0) {
#pragma unroll
        for (int f = 0; f < 16; ++f)
            a[f] = LD128(rx, xoff, r0 * (32 * KD * 2) + wave * 512 + (f >> 1) * 64 + (f & 1) * (16 * KD * 2), 0);
    } else {
#pragma unroll
        for (int f = 0; f < 16; ++f) asm volatile("" : "=v"(a[f]));
    }
    if (v == 0) {
        u32x2 raw[64];
#pragma unroll
        for (int n = 0; n < 64; ++n)
            raw[n] = LD64(rw, woff + q * 128 + (n & 7) * (NS * 2), (wave * 8 + (n >> 3)) * (32 * NS * 2));
#pragma unroll
        for (int j = 0; j < 8; ++j) {
#pragma unroll
            for (int n = 0; n < 4; ++n) bf[j][n] = __builtin_bit_cast(bf16x8, wfrag(raw + j * 8, n));
            asm volatile("" : "+v"(bf[j][0]), "+v"(bf[j][1]), "+v"(bf[j][2]), "+v"(bf[j][3]) :: "memory");
        }
    } else {
#pragma unroll
        for (int j = 0; j < 8; ++j) asm volatile("" : "=v"(bf[j][0]), "=v"(bf[j][1]), "=v"(bf[j][2]), "=v"(bf[j][3]));
    }
    GET_CALL();
    PUT_CALL();
    if (v | u0) {
        char* bbase = me + WS_RECV + slot * LINK + (v - 1) * LSTRIDE + (wave >> 2) * PH + (wave & 3) * 8192;
        char* abase = A_BASE(u0, r0);
        const auto rb = rsrc(bbase, PH - (wave & 3) * 8192), ra = rsrc(abase, PH - wave * 4096);
        const u64 all = (v ? 0x11111111ull : 0) | (u0 ? 0x1111ull << 32 : 0);
        char* can = (lane < 32 ? bbase + (lane >> 3) * 32768 + (lane & 7) * 1024 : abase + A_OFF(lane & 15)) + 1020;
        u64 todo = all, look = all * 15;
#pragma unroll 1
        for (;;) {
            const bool mine = (look >> lane) & 1;
            const u32 w = SYS_LOAD(mine ? can : me + WS_TMO);
            const u64 seen = __builtin_amdgcn_ballot_w64(mine && w != 0);
            u64 get = seen & (seen >> 1);
            get &= (get >> 2) & todo;
            if (ANY(!mine && w == call) || SINCE(t0) > DEADLINE) { ok = false; get = todo; }
#pragma unroll
            for (int G = 0; G < 8; ++G)
                if ((get >> (4 * G)) & 1) {
#pragma unroll
                    for (int i = 0; i < 4; ++i)
                        bf[(G & 1) * 4 + i][G >> 1] = __builtin_bit_cast(bf16x8, LD128(rb, lane16 + i * 1024, (G >> 1) * 32768 + (G & 1) * 4096, 17));
                }
#pragma unroll
            for (int G = 0; G < 4; ++G)
                if ((get >> (32 + 4 * G)) & 1) {
#pragma unroll
                    for (int i = 0; i < 4; ++i) a[G * 4 + i] = LD128(ra, lane16 + i * 1024, G * 32768, 17);
                }
            todo &= ~get;
            if (!todo) break;
            look = todo * 15;
            if (!seen) {
                if (todo == all) look = todo & 0x100000011ull;
                __builtin_amdgcn_s_sleep(POLL_SLEEP);
            }
        }
#pragma unroll 1
        for (;;) {
            u32 mb = ~0u, ma = ~0u;
#pragma unroll
            for (int f = 0; f < 32; ++f) mb = min4(mb, __builtin_bit_cast(u32x4, bf[f >> 2][f & 3]));
#pragma unroll
            for (int f = 0; f < 16; ++f) ma = min4(ma, a[f]);
            const bool rb_ = v && ANY(mb == 0), ra_ = u0 && ANY(ma == 0);
            if (__builtin_expect(!(rb_ || ra_) || !ok, 1)) break;
            if (rb_) {
#pragma unroll
                for (int f = 0; f < 32; ++f) bf[f >> 2][f & 3] = __builtin_bit_cast(bf16x8, LD128(rb, lane16, B_OFF(f), 17));
            }
            if (ra_) {
#pragma unroll
                for (int f = 0; f < 16; ++f) a[f] = LD128(ra, lane16, A_OFF(f), 17);
            }
            if (SINCE(t0) > DEADLINE) ok = false;
        }
    }
    asm volatile("" : "+s"(optr), "+s"(rank) :: "memory");
#pragma unroll
    for (int i = 0; i < 2; ++i)
#pragma unroll
        for (int n = 0; n < 4; ++n) acc[i][n] = f32x4{0.f, 0.f, 0.f, 0.f};
#pragma unroll
    for (int f = 0; f < 16; ++f) MMA(f);
    PUT(0);

    bool late = v == 0;
    if (!(td & D_SINGLE)) {
        char* abase = A_BASE(u1, r1);
        const u32 coff = A_OFF(lane & 15) + 1020;
        const u32x4 ra = {(u32)(u64)abase, (u32)((u64)abase >> 32) & 0xFFFFu, PH - wave * 4096, 0x00020000u};
        u32 seen = watch(abase + coff, me + WS_TMO, call, t0, lane, ok), pend = 0x1111, cur = 0, w = 0;
        u32 lim = ok ? 1 : 0;
#pragma unroll
        for (int i = 0; i < 2; ++i) {
#pragma unroll
            for (int n = 0; n < 4; ++n) acc[i][n] = f32x4{0.f, 0.f, 0.f, 0.f};
            asm volatile("" : "+v"(acc[i][0]), "+v"(acc[i][1]), "+v"(acc[i][2]), "+v"(acc[i][3]), "+s"(pend), "+s"(cur));
        }
#pragma unroll 1
        for (u32 it = 1;; ++it) {
            const u32 rest = pend & ~cur;
            u32 nxt = seen & (seen >> 1);
            nxt &= (nxt >> 2) & rest;
            if (__builtin_popcount(rest & ~nxt) <= SPEC) nxt = rest;
#pragma unroll
            for (int t = 0; t < 4; ++t)
                if ((nxt >> (4 * t)) & 1)
                    asm volatile("buffer_load_dwordx4 %0, %4, %5, %6 offen sc0 sc1\n\t"
                                 "buffer_load_dwordx4 %1, %4, %5, %6 offen offset:1024 sc0 sc1\n\t"
                                 "buffer_load_dwordx4 %2, %4, %5, %6 offen offset:2048 sc0 sc1\n\t"
                                 "buffer_load_dwordx4 %3, %4, %5, %6 offen offset:3072 sc0 sc1"
                                 : "+v"(a[4 * t]), "+v"(a[4 * t + 1]), "+v"(a[4 * t + 2]), "+v"(a[4 * t + 3])
                                 : "v"(lane16), "s"(ra), "s"(t * 32768) : "memory");
            if (rest != nxt)
                asm volatile("buffer_load_dword %0, %1, %2, 0 offen sc0 sc1" : "+v"(w) : "v"(coff), "s"(ra) : "memory");
#pragma unroll
            for (int t = 0; t < 4; ++t)
                if ((cur >> (4 * t)) & 1) {
                    u32 m = ~0u;
#pragma unroll
                    for (int f = 0; f < 4; ++f) m = min4(m, a[4 * t + f]);
                    if (!ANY(m < lim)) {
                        MMA(4 * t); MMA(4 * t + 1); MMA(4 * t + 2); MMA(4 * t + 3);
                        pend &= ~(1u << (4 * t));
                    }
                }
            if (!pend) break;
            if (!late) late = SINCE(t0) >= T_Y;
            cur = nxt;
            asm volatile("s_waitcnt vmcnt(0)"
                         : "+v"(a[0]), "+v"(a[1]), "+v"(a[2]), "+v"(a[3]), "+v"(a[4]), "+v"(a[5]), "+v"(a[6]), "+v"(a[7]),
                           "+v"(a[8]), "+v"(a[9]), "+v"(a[10]), "+v"(a[11]), "+v"(a[12]), "+v"(a[13]), "+v"(a[14]),
                           "+v"(a[15]), "+v"(w) :: "memory");
            seen = (u32)__builtin_amdgcn_ballot_w64(w != 0);
            if ((it & 255) == 0 && SINCE(t0) > DEADLINE) { ok = false; lim = 0; seen = 0xFFFF; }
        }
        PUT(1);
    }
    __syncthreads();
    const int h = wave < 4;
    if (!(h && (td & D_SINGLE))) {
        const int t = tid & 255;
        const f32x4* rs = red[h] + t;
        f32x4 s0 = rs[0], s1 = rs[RED_H];
#pragma unroll
        for (int n = 1; n < 8; ++n) { s0 += rs[n * RED_W]; s1 += rs[n * RED_W + RED_H]; }
        typedef __bf16 bf16x4_t __attribute__((ext_vector_type(4)));
        const u32x2 o0 = __builtin_bit_cast(u32x2, __builtin_convertvector(s0, bf16x4_t));
        const u32x2 o1 = __builtin_bit_cast(u32x2, __builtin_convertvector(s1, bf16x4_t));
        u32x4 o = {o0[0], o0[1], o1[0], o1[1]};
        const u32 ooff = (((rank ^ (h ? u1 : u0)) & 7) * MS + (h ? r1 : r0) * 32 + (t >> 3)) * (NS * 2) + q * 128 + (t & 7) * 16;
        if (v) o = nz(o);
        if (!late) while (SINCE(t0) < T_Y) __builtin_amdgcn_s_sleep(2);
        __builtin_amdgcn_raw_buffer_store_b128(o, rsrc(optr, SHARD_BYTES), ooff, 0, 17);
    }
}

static void build_table(u32* tab) {
    u32 list[8][32];
    int n[8] = {0};
    auto add = [&](int x, u32 d) { list[x][n[x]++] = d; };
    auto ur = [](int u, int r) { return (u32)(u | r << 3); };
    auto pair = [](const u32* e, int m, int i) { return 2 * i + 1 == m ? (e[i] | D_SINGLE) : (e[i] | e[(m + 1) / 2 + i] << 6); };
    for (int x = 1; x < 8; ++x)
        for (int i = 0; i < 16; ++i) add(x, D_HELP | D_PUSH | i);
    for (int i = 0; i < NCLEAN; ++i) add(0, D_HELP | i);
    const int S[4][5][2] = {{{0, 0}, {0, 1}, {1, 2}, {1, 4}, {2, 4}}, {{0, 2}, {1, 0}, {1, 1}, {1, 3}, {3, 4}},
                            {{0, 3}, {2, 0}, {2, 1}, {2, 2}, {2, 3}}, {{0, 4}, {3, 0}, {3, 1}, {3, 2}, {3, 3}}};
    for (int x = 4; x < 8; ++x)
        for (int v = 4; v < 8; ++v) {
            u32 e[8];
            int m = 0;
            for (int ph = -1; ph < 5; ++ph) {
                for (int i = 0; i < 5; ++i) {
                    const int u = S[x - 4][i][0], r = S[x - 4][i][1];
                    if ((u ? r : -1) == ph) e[m++] = ur(u, r);
                }
                if (ph >= 2 && x != v) e[m++] = ur(x, ph + 3);
            }
            for (int i = 0; i < (m + 1) / 2; ++i) add(x, pair(e, m, i) | v << 12 | (v & 3) << 15);
        }
    {
        u32 e[35];
        int m = 0;
        for (int r = 0; r < 8; ++r) e[m++] = ur(0, r);
        for (int ph = 0; ph < 5; ++ph) {
            for (int u = 1; u < 4; ++u) e[m++] = ur(u, ph);
            if (ph >= 2) for (int u = 4; u < 8; ++u) e[m++] = ur(u, ph + 3);
        }
        for (int i = 0; i < 18; ++i)
            for (int q = 0; q < 4; ++q) add(i == 17 ? 4 + q : i == 16 ? 0 : i & 3, pair(e, m, i) | q << 15);
    }
    for (int x = 0; x < 8; ++x)
        for (int i = 0; i < 32; ++i) tab[i * 8 + x] = list[x][((i >> 2) & 1) * 16 + ((i >> 3) << 2) + (i & 3)];
}

extern "C" void kernel_launch(int rank, int nranks, ncclComm_t comm, hipStream_t stream,
                              void* const* d_in, const int* in_sizes, int n_in,
                              void* d_out, int out_size, const int* in_dtypes, int out_dtype,
                              void* const* d_out_all, void* const* d_ws_all, size_t ws_bytes) {
    (void)comm; (void)in_dtypes; (void)out_dtype; (void)d_out;
    if (nranks != NR || n_in != 2 || in_sizes[0] != MS * KD || in_sizes[1] != KD * NS ||
        out_size != NR * MS * NS || ws_bytes < WS_END || (unsigned)rank >= (unsigned)NR) {
        fprintf(stderr, "unexpected problem shape\n");
        return;
    }
    Params p;
    p.x = d_in[0];
    p.w = d_in[1];
    for (int i = 0; i < NR; ++i) {
        p.out[i] = (char*)d_out_all[rank ^ i];
        p.ws[i] = (char*)d_ws_all[rank ^ i];
    }
    p.rank = rank;
    build_table(p.tab);
    ag_gemm<<<256, 512, 0, stream>>>(p);
    HIP_CHECK(hipGetLastError());
}
```
